```python
import numpy as np

kernel_name = "gpu_gemm_a2a_m4096_k4096_n32768_f32_none"


ABSMAX_RTOL = 0.001

COMPUTE_DTYPE = "float32"
EPILOGUE = "none"
IN_QUANT = None
OUT_QUANT = None
IN_DTYPES = {"x": "float32", "w_mat": "float32"}
WORLD_SIZE = 8

def setup_inputs():

    x = np.random.randn(4096, 4096).astype(np.float32)
    w_mat = (np.random.randn(4096, 32768) / np.sqrt(4096)).astype(np.float32)
    return {"x": x, "w_mat": w_mat}

def _epilogue(y):
    return y

def reference(x, w_mat):
    y = (x @ w_mat).astype(np.float32)
    return _epilogue(y)
```

```cpp
#include <hip/hip_runtime.h>
#include <rccl/rccl.h>
#include <cstdio>

namespace {
#define LAS __attribute__((address_space(3)))
typedef _Float16 h8 __attribute__((ext_vector_type(8)));
typedef _Float16 h2 __attribute__((ext_vector_type(2)));
typedef float f4 __attribute__((ext_vector_type(4)));

constexpr int R = 8, M = 4096, K = 4096, NL = 4096, NG = 32768, MS = 512;
constexpr int BM = 256, BK = 64, HALF = 128, HTB = HALF * BK * 2, STAGE_BYTES = 8 * HTB;
constexpr int NWG = 256, NTHR = 512;
constexpr size_t OFF_X = 1ull << 20, XBYTES = (size_t)M * K * 2, OFF_WT = OFF_X + 2 * XBYTES;
constexpr int CTL_TICKET0 = 0, CTL_TICKET1 = 128, CTL_TMO = 256, CTL_XCNT = 4096;
constexpr unsigned long long DEADLINE_TICKS = 40000000ull;

__device__ __forceinline__ int lds_byte(int r, int c) {
  const int st = (r >> 4) * 2 + (c >> 5), rr = r & 15, cc = c & 31, ob = rr * 64 + cc * 2;
  return st * 1024 + (ob ^ (((ob >> 9) & 1) << 5));
}
__device__ __forceinline__ void stage_rc(int b, int& Rr, int& Cc) {
  const int st = b / 1024, sb = b % 1024, swz = sb ^ (((sb >> 9) & 1) << 5);
  Rr = (st >> 1) * 16 + swz / 64;
  Cc = (st & 1) * 32 + (swz % 64) / 2;
}

struct Args {
  const float* x;
  const float* w;
  float* out;
  void* const* ws_all;
  int rank;
  int phase;
};

__device__ __forceinline__ unsigned draw_call(unsigned* ctr, unsigned* sh) {
  if (threadIdx.x == 0) {
    unsigned t = __hip_atomic_fetch_add(ctr, 1u, __ATOMIC_RELAXED, __HIP_MEMORY_SCOPE_AGENT);
    *sh = t / gridDim.x + 1u;
  }
  __syncthreads();
  unsigned c = *sh;
  __syncthreads();
  return c;
}

__device__ void phase0(const Args& a, unsigned char* lds) {
  char* myws = (char*)a.ws_all[a.rank];
  const unsigned call = draw_call((unsigned*)(myws + CTL_TICKET0), (unsigned*)lds);
  const int tid = threadIdx.x, w = blockIdx.x;
  const size_t xoff = OFF_X + (size_t)(call & 1) * XBYTES;
  {
    const int row = 2 * w + (tid >> 8), col = (tid & 255) * 16;
    const f4* src = (const f4*)(a.x + (size_t)row * K + col);
    f4 v0 = src[0], v1 = src[1], v2 = src[2], v3 = src[3];
    h8 o0, o1;
#pragma unroll
    for (int j = 0; j < 4; ++j) {
      o0[j] = (_Float16)v0[j]; o0[4 + j] = (_Float16)v1[j];
      o1[j] = (_Float16)v2[j]; o1[4 + j] = (_Float16)v3[j];
    }
    const size_t doff = xoff + ((size_t)(a.rank * MS + row) * K + col) * 2;
#pragma unroll
    for (int i = 0; i < R; ++i) {
      const unsigned p = (unsigned)(a.rank + i) & (R - 1);
      h8* dst = (h8*)((char*)a.ws_all[p] + doff);
      dst[0] = o0; dst[1] = o1;
    }
  }
  {
    const int wid = tid >> 6, lane = tid & 63;
    _Float16* wt = (_Float16*)(myws + OFF_WT);
    unsigned char* tl = lds + 64 + wid * (64 * 136);
    for (int it = 0; it < 2; ++it) {
      const int tile = (w * 8 + wid) * 2 + it;
      const int k0 = (tile >> 6) * 64, n0 = (tile & 63) * 64;
      const int c4 = lane & 15, q = lane >> 4;
      const float* src = a.w + (size_t)k0 * NG + (size_t)a.rank * NL + n0 + c4 * 4;
      f4 v[16];
#pragma unroll
      for (int i = 0; i < 8; ++i) {
        v[2 * i] = *(const f4*)(src + (size_t)(i * 8 + q * 2) * NG);
        v[2 * i + 1] = *(const f4*)(src + (size_t)(i * 8 + q * 2 + 1) * NG);
      }
#pragma unroll
      for (int i = 0; i < 8; ++i)
#pragma unroll
        for (int j = 0; j < 4; ++j) {
          h2 p; p[0] = (_Float16)v[2 * i][j]; p[1] = (_Float16)v[2 * i + 1][j];
          *(h2*)(tl + (c4 * 4 + j) * 136 + (i * 8 + q * 2) * 2) = p;
        }
      __builtin_amdgcn_s_waitcnt(0xc07f);
      __builtin_amdgcn_wave_barrier();
#pragma unroll
      for (int i = 0; i < 8; ++i) {
        const int n = i * 8 + (lane >> 3), k8 = lane & 7;
        typedef unsigned long long u64;
        const u64* s = (const u64*)(tl + n * 136 + k8 * 16);
        u64 lo = s[0], hi = s[1];
        u64* d = (u64*)(wt + (size_t)(n0 + n) * K + k0 + k8 * 8);
        d[0] = lo; d[1] = hi;
      }
      __builtin_amdgcn_s_waitcnt(0xc07f);
      __builtin_amdgcn_wave_barrier();
    }
  }
  asm volatile("s_waitcnt vmcnt(0)" ::: "memory");
  __syncthreads();
  if (tid < R) {
    __builtin_amdgcn_fence(__ATOMIC_RELEASE, "");
    asm volatile("s_waitcnt vmcnt(0)" ::: "memory");
    unsigned* cnt = (unsigned*)((char*)a.ws_all[tid] + CTL_XCNT + a.rank * 128);
    __hip_atomic_fetch_add(cnt, 1u, __ATOMIC_RELAXED, __HIP_MEMORY_SCOPE_SYSTEM);
  }
}

__device__ void phase1(const Args& a, LAS unsigned char* lds, unsigned char* glds) {
  char* myws = (char*)a.ws_all[a.rank];
  const unsigned call = draw_call((unsigned*)(myws + CTL_TICKET1), (unsigned*)glds);
  const int tid = threadIdx.x, wid = __builtin_amdgcn_readfirstlane(tid >> 6), lane = tid & 63;
  const int wr = wid >> 2, wc = wid & 3, fr = lane & 15, fq = lane >> 4;
  unsigned* tmo = (unsigned*)(myws + CTL_TMO);
  if (tid == 0) {
    const unsigned long long t0 = __builtin_amdgcn_s_memrealtime();
    for (int p = 0; p < R; ++p) {
      const unsigned* cnt = (const unsigned*)(myws + CTL_XCNT + p * 128);
      while (__hip_atomic_load(cnt, __ATOMIC_RELAXED, __HIP_MEMORY_SCOPE_SYSTEM) < call * (unsigned)NWG) {
        if (__hip_atomic_load(tmo, __ATOMIC_RELAXED, __HIP_MEMORY_SCOPE_AGENT) == call ||
            __builtin_amdgcn_s_memrealtime() - t0 > DEADLINE_TICKS) {
          __hip_atomic_store(tmo, call, __ATOMIC_RELAXED, __HIP_MEMORY_SCOPE_AGENT);
          break;
        }
        __builtin_amdgcn_s_sleep(2);
      }
    }
    __builtin_amdgcn_fence(__ATOMIC_ACQUIRE, "");
    asm volatile("s_waitcnt vmcnt(0)" ::: "memory");
  }
  __syncthreads();
  const bool bad = __hip_atomic_load(tmo, __ATOMIC_RELAXED, __HIP_MEMORY_SCOPE_AGENT) == call;

  int wg = blockIdx.x;
  wg = (wg & 7) * 32 + (wg >> 3);
  const int pm = (wg >> 5) * 2 + ((wg >> 4) & 1) * 0 + (wg & 1);
  const int pn = (wg & 31) >> 1;
  const char* Ab = myws + OFF_X + (size_t)(call & 1) * XBYTES;
  const char* Bb = myws + OFF_WT;

  unsigned voff[2];
#pragma unroll
  for (int i = 0; i < 2; ++i) { int Rr, Cc; stage_rc(tid * 16 + i * 8192, Rr, Cc); voff[i] = (unsigned)(Rr * K + Cc) * 2u; }
  const size_t kstep = (size_t)(BK * 2), hstep = (size_t)HALF * K * 2, tstep = 2 * hstep;
  const unsigned ldsw = (unsigned)wid * 1024u;
  const int aoff = lds_byte(wr * 64 + fr, fq * 8), boff = lds_byte(wc * 32 + fr, fq * 8);
#define SA(b, h) (((b) * 2 + (h)) * HTB)
#define SB(b, h) ((4 + (b) * 2 + (h)) * HTB)
#define STAGE(bufoff, gbase) do { _Pragma("unroll") for (int _i = 0; _i < 2; ++_i) \
    __builtin_amdgcn_global_load_lds((const unsigned*)((const char*)(gbase) + voff[_i]), (LAS unsigned*)(lds + (bufoff) + ldsw + _i * 8192), 16, 0, 0); } while (0)
#define LDA(dst, b, h) do { _Pragma("unroll") for (int m = 0; m < 4; ++m) _Pragma("unroll") for (int k = 0; k < 2; ++k) dst[m][k] = *(const LAS h8*)(lds + SA(b, h) + aoff + m * 2048 + k * 1024); } while (0)
#define LDB(dst, b, h) do { _Pragma("unroll") for (int n = 0; n < 2; ++n) _Pragma("unroll") for (int k = 0; k < 2; ++k) dst[n][k] = *(const LAS h8*)(lds + SB(b, h) + boff + n * 2048 + k * 1024); } while (0)
#define MMA(ai, bj, At, Bt) do { __builtin_amdgcn_s_setprio(1); _Pragma("unroll") for (int m = 0; m < 4; ++m) _Pragma("unroll") for (int n = 0; n < 2; ++n) _Pragma("unroll") for (int k = 0; k < 2; ++k) \
    acc[ai][bj][m][n] = __builtin_amdgcn_mfma_f32_16x16x32_f16(Bt[n][k], At[m][k], acc[ai][bj][m][n], 0, 0, 0); __builtin_amdgcn_s_setprio(0); } while (0)
#define WAIT_V(n) asm volatile("s_waitcnt vmcnt(" #n ")" ::: "memory")
#define WAIT_L(n) asm volatile("s_waitcnt lgkmcnt(" #n ")" ::: "memory")
#define BAR __builtin_amdgcn_s_barrier()
#define SCHED __builtin_amdgcn_sched_barrier(0)
  f4 acc[2][2][4][2];
#pragma unroll
  for (int i = 0; i < 2; ++i)
#pragma unroll
    for (int j = 0; j < 2; ++j)
#pragma unroll
      for (int m = 0; m < 4; ++m)
#pragma unroll
        for (int n = 0; n < 2; ++n) acc[i][j][m][n] = (f4){0.f, 0.f, 0.f, 0.f};
  h8 At[4][2], B0[2][2], B1[2][2];
  const char* cA = Ab + (size_t)pm * tstep;
  const char* cB = Bb + (size_t)pn * tstep;
  constexpr int nt = K / BK;
  STAGE(SB(0, 0), cB); STAGE(SA(0, 0), cA); STAGE(SB(0, 1), cB + hstep); STAGE(SA(0, 1), cA + hstep);
  if (wr == 1) BAR;
  WAIT_V(4); BAR;
  STAGE(SB(1, 0), cB + kstep); STAGE(SA(1, 0), cA + kstep); STAGE(SB(1, 1), cB + hstep + kstep);
  WAIT_V(6); BAR;
  for (int t = 0; t < nt; t += 2) {
    const bool last = (t == nt - 2);
    const char* a1 = cA + (size_t)(t + 1) * kstep;
    const char* a2 = last ? cA : cA + (size_t)(t + 2) * kstep;
    const char* b2 = last ? cB : cB + (size_t)(t + 2) * kstep;
    const char* a3 = a2 + kstep;
    const char* b3 = b2 + kstep;
    LDB(B0, 0, 0); SCHED; LDA(At, 0, 0); STAGE(SA(1, 1), a1 + hstep);
    WAIT_L(8); BAR; WAIT_L(0); MMA(0, 0, At, B0); BAR; SCHED;
    LDB(B1, 0, 1); STAGE(SB(0, 0), b2);
    BAR; WAIT_L(0); MMA(0, 1, At, B1); BAR;
    LDA(At, 0, 1); STAGE(SA(0, 0), a2);
    BAR; WAIT_L(0); MMA(1, 0, At, B0); BAR; SCHED;
    STAGE(SB(0, 1), b2 + hstep);
    WAIT_V(6); BAR; MMA(1, 1, At, B1); BAR;
    LDB(B0, 1, 0); SCHED; LDA(At, 1, 0); STAGE(SA(0, 1), a2 + hstep);
    WAIT_L(8); BAR; WAIT_L(0); MMA(0, 0, At, B0); BAR; SCHED;
    LDB(B1, 1, 1); STAGE(SB(1, 0), b3);
    BAR; WAIT_L(0); MMA(0, 1, At, B1); BAR;
    LDA(At, 1, 1); STAGE(SA(1, 0), a3);
    BAR; WAIT_L(0); MMA(1, 0, At, B0); BAR; SCHED;
    STAGE(SB(1, 1), b3 + hstep);
    WAIT_V(6); BAR; MMA(1, 1, At, B1); BAR;
  }
  {
    const float nanv = __builtin_nanf("");
    const int row0 = pm * BM + wr * 64 + fr, col0 = pn * BM + wc * 32 + 4 * fq;
#pragma unroll
    for (int ai = 0; ai < 2; ++ai)
#pragma unroll
      for (int m = 0; m < 4; ++m) {
        float* rowp = a.out + (size_t)(row0 + ai * HALF + m * 16) * NL + col0;
#pragma unroll
        for (int bj = 0; bj < 2; ++bj)
#pragma unroll
          for (int n = 0; n < 2; ++n)
            *(f4*)(rowp + bj * HALF + n * 16) = bad ? (f4){nanv, nanv, nanv, nanv} : acc[ai][bj][m][n];
      }
  }
  WAIT_V(0);
  if (wr == 0) BAR;
  BAR;
}

__global__ void __launch_bounds__(NTHR, 2) fused_kernel(Args a) {
  extern __shared__ __attribute__((aligned(16))) unsigned char shm[];
  if (a.phase == 0) phase0(a, shm);
  else phase1(a, (LAS unsigned char*)shm, shm);
}
}

extern "C" void kernel_launch(int rank, int nranks, ncclComm_t comm, hipStream_t stream,
                              void* const* d_in, const int* in_sizes, int n_in, void* d_out, int out_size,
                              const int* in_dtypes, int out_dtype, void* const* d_out_all,
                              void* const* d_ws_all, size_t ws_bytes) {
  static bool init = false;
  if (!init) {
    hipFuncSetAttribute((const void*)fused_kernel, hipFuncAttributeMaxDynamicSharedMemorySize, STAGE_BYTES);
    init = true;
  }
  Args a;
  a.x = (const float*)d_in[0];
  a.w = (const float*)d_in[1];
  a.out = (float*)d_out;
  a.ws_all = d_ws_all;
  a.rank = rank;
  a.phase = 0;
  hipLaunchKernelGGL(fused_kernel, dim3(NWG), dim3(NTHR), STAGE_BYTES, stream, a);
  a.phase = 1;
  hipLaunchKernelGGL(fused_kernel, dim3(NWG), dim3(NTHR), STAGE_BYTES, stream, a);
}
```

```cpp
#include <hip/hip_runtime.h>
#include <rccl/rccl.h>
#include <cstdio>

namespace {
#define LAS __attribute__((address_space(3)))
typedef _Float16 h8 __attribute__((ext_vector_type(8)));
typedef _Float16 h2 __attribute__((ext_vector_type(2)));
typedef float f4 __attribute__((ext_vector_type(4)));

constexpr int R = 8, M = 4096, K = 4096, NL = 4096, NG = 32768, MS = 512;
constexpr int BM = 256, BK = 64, HALF = 128, HTB = HALF * BK * 2, STAGE_BYTES = 8 * HTB;
constexpr int NWG = 256, NTHR = 512, NT = K / BK;
constexpr int LEAD = 16, EARLY = 16;
constexpr size_t TILE_BYTES = (size_t)BM * BK * 2;
constexpr int WUNITS = 512;
constexpr size_t OFF_X = 1ull << 20, XBYTES = (size_t)M * K * 2, OFF_WT = OFF_X + XBYTES;
constexpr int NCH = 4, CH_TILES = NT / NCH;
constexpr size_t CH_BYTES = 2 * CH_TILES * (size_t)BM * BK * 2;
constexpr int CTL_TICKET = 0, CTL_TMO = 256, CTL_TICKET1 = 512, CTL_SEQ = 768, CTL_GDONE = 1024, CTL_PDONE = 2048, CTL_FLAGS = 4096;
constexpr unsigned long long DEADLINE_TICKS = 40000000ull;

__device__ __forceinline__ int lds_byte(int r, int c) {
  const int st = (r >> 4) * 2 + (c >> 5), rr = r & 15, cc = c & 31, ob = rr * 64 + cc * 2;
  return st * 1024 + (ob ^ (((ob >> 9) & 1) << 5));
}
__device__ __forceinline__ void stage_rc(int b, int& Rr, int& Cc) {
  const int st = b / 1024, sb = b % 1024, swz = sb ^ (((sb >> 9) & 1) << 5);
  Rr = (st >> 1) * 16 + swz / 64;
  Cc = (st & 1) * 32 + (swz % 64) / 2;
}

struct Args {
  const float* x;
  const float* w;
  float* out;
  void* const* ws_all;
  int rank;
  int phase;
};

#define stamp(a, slot)

__device__ __forceinline__ void load_piece(const Args& a, int kc, int h, h8 (&o)[4]) {
  const int tid = threadIdx.x;
  const int r0 = h * 256 + (tid >> 3), c = kc * 64 + (tid & 7) * 8;
  f4 v[8];
#pragma unroll
  for (int i = 0; i < 4; ++i) {
    const f4* src = (const f4*)(a.x + (size_t)(r0 + i * 64) * K + c);
    v[2 * i] = src[0]; v[2 * i + 1] = src[1];
  }
#pragma unroll
  for (int i = 0; i < 4; ++i)
#pragma unroll
    for (int j = 0; j < 4; ++j) { o[i][j] = (_Float16)v[2 * i][j]; o[i][4 + j] = (_Float16)v[2 * i + 1][j]; }
}

__device__ __forceinline__ size_t xtile_off(int pm, int kc) {
  return (size_t)(pm >> 1) * (NCH * CH_BYTES) + (size_t)(kc / CH_TILES) * CH_BYTES +
         (size_t)((pm & 1) * CH_TILES + (kc % CH_TILES)) * TILE_BYTES;
}

__device__ __forceinline__ void load_wtile(const Args& a, int tile, f4 (&v)[16]) {
  const int lane = threadIdx.x & 63;
  const int k0 = (tile >> 6) * 64, n0 = (tile & 63) * 64;
  const int c4 = lane & 15, q = lane >> 4;
  const float* src = a.w + (size_t)k0 * NG + (size_t)a.rank * NL + n0 + c4 * 4;
#pragma unroll
  for (int i = 0; i < 8; ++i) {
    v[2 * i] = *(const f4*)(src + (size_t)(i * 8 + q * 2) * NG);
    v[2 * i + 1] = *(const f4*)(src + (size_t)(i * 8 + q * 2 + 1) * NG);
  }
}

__device__ __forceinline__ void store_wtile(_Float16* wt, unsigned char* tl, int tile, const f4 (&v)[16]) {
  const int lane = threadIdx.x & 63;
  const int k0 = (tile >> 6) * 64, n0 = (tile & 63) * 64;
  const int c4 = lane & 15, q = lane >> 4;
#pragma unroll
  for (int i = 0; i < 8; ++i)
#pragma unroll
    for (int j = 0; j < 4; ++j) {
      h2 p; p[0] = (_Float16)v[2 * i][j]; p[1] = (_Float16)v[2 * i + 1][j];
      *(h2*)(tl + (c4 * 4 + j) * 136 + (i * 8 + q * 2) * 2) = p;
    }
  __builtin_amdgcn_s_waitcnt(0xc07f);
  __builtin_amdgcn_wave_barrier();
#pragma unroll
  for (int i = 0; i < 8; ++i) {
    const int n = i * 8 + (lane >> 3), k8 = lane & 7;
    typedef unsigned long long u64;
    typedef u64 u64x2 __attribute__((ext_vector_type(2)));
    const u64* s = (const u64*)(tl + n * 136 + k8 * 16);
    u64x2 v2; v2[0] = s[0]; v2[1] = s[1];
    *(u64x2*)((char*)wt + (size_t)((n0 >> 8) * NT + (k0 >> 6)) * TILE_BYTES + ((n0 & 255) + n) * 128 + k8 * 16) = v2;
  }
  __builtin_amdgcn_s_waitcnt(0xc07f);
  __builtin_amdgcn_wave_barrier();
}

__device__ __forceinline__ bool timed_out(unsigned* tmo, unsigned call, unsigned long long t0) {
  if (__hip_atomic_load(tmo, __ATOMIC_RELAXED, __HIP_MEMORY_SCOPE_AGENT) == call ||
      __builtin_amdgcn_s_memrealtime() - t0 > DEADLINE_TICKS) {
    __hip_atomic_store(tmo, call, __ATOMIC_RELAXED, __HIP_MEMORY_SCOPE_AGENT);
    return true;
  }
  return false;
}

__device__ __forceinline__ int wait_chunks(const unsigned* flags, unsigned* tmo, unsigned call, int need) {
  const int lane = threadIdx.x & 63;
  const unsigned long long t0 = __builtin_amdgcn_s_memrealtime();
  int ready;
  for (;;) {
    const unsigned f = lane < NCH ? __hip_atomic_load(flags + lane, __ATOMIC_RELAXED, __HIP_MEMORY_SCOPE_SYSTEM) : 0u;
    const unsigned long long m = __ballot(f >= call);
    ready = __builtin_ctzll(~m);
    if (ready > need) break;
    if (timed_out(tmo, call, t0)) { ready = NCH; break; }
    __builtin_amdgcn_s_sleep(2);
  }
  asm volatile("" ::: "memory");
  return ready;
}

__global__ void __launch_bounds__(NTHR, 2) fused_kernel(Args a) {
  extern __shared__ __attribute__((aligned(16))) unsigned char shm[];
  LAS unsigned char* lds = (LAS unsigned char*)shm;
  stamp(a, 0);
  char* myws = (char*)a.ws_all[a.rank];
  unsigned* tmo = (unsigned*)(myws + CTL_TMO);
  const int tid = threadIdx.x, wid = __builtin_amdgcn_readfirstlane(tid >> 6), lane = tid & 63;

  unsigned call;
  {
    unsigned* sh = (unsigned*)shm;
    if (tid == 0) {
      const unsigned t = __hip_atomic_fetch_add((unsigned*)(myws + (a.phase ? CTL_TICKET1 : CTL_TICKET)), 1u, __ATOMIC_RELAXED, __HIP_MEMORY_SCOPE_AGENT);
      sh[0] = t / gridDim.x + 1u;
      __builtin_amdgcn_fence(__ATOMIC_ACQUIRE, "");
      asm volatile("s_waitcnt vmcnt(0)" ::: "memory");
    }
    __syncthreads();
    call = sh[0];
    __syncthreads();
  }

  if (a.phase == 0) {
    _Float16* wt = (_Float16*)(myws + OFF_WT);
    unsigned char* tl = shm + 64 + wid * (64 * 136);
    const int tile0 = (int)blockIdx.x * 16 + wid * 2;
    f4 va[16], vb[16];
    load_wtile(a, tile0, va);
    load_wtile(a, tile0 + 1, vb);
    if (blockIdx.x < 128) {
      if (tid < R && tid != a.rank) {
        const unsigned* pd = (const unsigned*)(myws + CTL_PDONE) + tid * 32;
        const unsigned long long t0 = __builtin_amdgcn_s_memrealtime();
        while (__hip_atomic_load(pd, __ATOMIC_RELAXED, __HIP_MEMORY_SCOPE_SYSTEM) + 1u < call) {
          if (timed_out(tmo, call, t0)) break;
          __builtin_amdgcn_s_sleep(2);
        }
      }
      const int kc = (int)blockIdx.x >> 1, h = (int)blockIdx.x & 1;
      h8 eo[4];
      load_piece(a, kc, h, eo);
      char* dst = myws + OFF_X + xtile_off(a.rank * 2 + h, kc) + tid * 16;
#pragma unroll
      for (int j = 0; j < 4; ++j) *(h8*)(dst + j * 8192) = eo[j];
      if (blockIdx.x == 0 && tid == 0) *(unsigned*)(myws + CTL_SEQ) = call;
    }
    store_wtile(wt, tl, tile0, va);
    store_wtile(wt, tl, tile0 + 1, vb);
    stamp(a, 1);
    __syncthreads();
    return;
  }
  stamp(a, 2);

  const int wr = wid >> 2, wc = wid & 3, fr = lane & 15, fq = lane >> 4;
  int wg = blockIdx.x;
  wg = (wg & 7) * 32 + (wg >> 3);
  const int pm = (wg >> 5) * 2 + (wg & 1);
  const int pn = (wg & 31) >> 1;
  const char* Ab = myws + OFF_X;
  const char* Bb = myws + OFF_WT;
  const unsigned* flags = (const unsigned*)(myws + CTL_FLAGS) + (pm >> 1) * 32;

  unsigned voff[2];
#pragma unroll
  for (int i = 0; i < 2; ++i) { int Rr, Cc; stage_rc(tid * 16 + i * 8192, Rr, Cc); voff[i] = (unsigned)(Rr * BK + Cc) * 2u; }
  const size_t kstep = TILE_BYTES, hstep = TILE_BYTES / 2, tstep = TILE_BYTES * NT;
  const unsigned ldsw = (unsigned)wid * 1024u;
  const int aoff = lds_byte(wr * 64 + fr, fq * 8), boff = lds_byte(wc * 32 + fr, fq * 8);
#define SA(b, h) (((b) * 2 + (h)) * HTB)
#define SB(b, h) ((4 + (b) * 2 + (h)) * HTB)
#define STAGE(bufoff, gbase) do { _Pragma("unroll") for (int _i = 0; _i < 2; ++_i) \
    __builtin_amdgcn_global_load_lds((const unsigned*)((const char*)(gbase) + voff[_i]), (LAS unsigned*)(lds + (bufoff) + ldsw + _i * 8192), 16, 0, 0); } while (0)
#define LDA(dst, b, h) do { _Pragma("unroll") for (int m = 0; m < 4; ++m) _Pragma("unroll") for (int k = 0; k < 2; ++k) dst[m][k] = *(const LAS h8*)(lds + SA(b, h) + aoff + m * 2048 + k * 1024); } while (0)
#define LDB(dst, b, h) do { _Pragma("unroll") for (int n = 0; n < 2; ++n) _Pragma("unroll") for (int k = 0; k < 2; ++k) dst[n][k] = *(const LAS h8*)(lds + SB(b, h) + boff + n * 2048 + k * 1024); } while (0)
#define MMA(ai, bj, At, Bt) do { __builtin_amdgcn_s_setprio(1); _Pragma("unroll") for (int m = 0; m < 4; ++m) _Pragma("unroll") for (int n = 0; n < 2; ++n) _Pragma("unroll") for (int k = 0; k < 2; ++k) \
    acc[ai][bj][m][n] = __builtin_amdgcn_mfma_f32_16x16x32_f16(Bt[n][k], At[m][k], acc[ai][bj][m][n], 0, 0, 0); __builtin_amdgcn_s_setprio(0); } while (0)
#define WAIT_V(n) asm volatile("s_waitcnt vmcnt(" #n ")" ::: "memory")
#define WAIT_L(n) asm volatile("s_waitcnt lgkmcnt(" #n ")" ::: "memory")
#define BAR __builtin_amdgcn_s_barrier()
#define SCHED __builtin_amdgcn_sched_barrier(0)
  f4 acc[2][2][4][2];
#pragma unroll
  for (int i = 0; i < 2; ++i)
#pragma unroll
    for (int j = 0; j < 2; ++j)
#pragma unroll
      for (int m = 0; m < 4; ++m)
#pragma unroll
        for (int n = 0; n < 2; ++n) acc[i][j][m][n] = (f4){0.f, 0.f, 0.f, 0.f};
  h8 At[4][2], B0[2][2], B1[2][2];
  const char* cA = Ab + xtile_off(pm, 0);
  const char* cB = Bb + (size_t)pn * tstep;
  int ready = (pm >> 1) == a.rank ? NCH : wait_chunks(flags, tmo, call, 0);
  stamp(a, 3);
  STAGE(SB(0, 0), cB); STAGE(SA(0, 0), cA); STAGE(SB(0, 1), cB + hstep); STAGE(SA(0, 1), cA + hstep);
  if (wr == 1) BAR;
  WAIT_V(4); BAR;
  STAGE(SB(1, 0), cB + kstep); STAGE(SA(1, 0), cA + kstep); STAGE(SB(1, 1), cB + hstep + kstep);
  WAIT_V(6); BAR;
  for (int t = 0; t < NT; t += 2) {
    const bool last = (t == NT - 2);
    const char* a1 = Ab + xtile_off(pm, t + 1);
    const char* a2 = last ? cA : Ab + xtile_off(pm, t + 2);
    const char* b2 = last ? cB : cB + (size_t)(t + 2) * kstep;
    const char* a3 = a2 + kstep;
    const char* b3 = b2 + kstep;
    if (!last && ready <= (t + 3) / CH_TILES) ready = wait_chunks(flags, tmo, call, (t + 3) / CH_TILES);
    LDB(B0, 0, 0); SCHED; LDA(At, 0, 0); STAGE(SA(1, 1), a1 + hstep);
    WAIT_L(8); BAR; WAIT_L(0); MMA(0, 0, At, B0); BAR; SCHED;
    LDB(B1, 0, 1); STAGE(SB(0, 0), b2);
    BAR; WAIT_L(0); MMA(0, 1, At, B1); BAR;
    LDA(At, 0, 1); STAGE(SA(0, 0), a2);
    BAR; WAIT_L(0); MMA(1, 0, At, B0); BAR; SCHED;
    STAGE(SB(0, 1), b2 + hstep);
    WAIT_V(6); BAR; MMA(1, 1, At, B1); BAR;
    LDB(B0, 1, 0); SCHED; LDA(At, 1, 0); STAGE(SA(0, 1), a2 + hstep);
    WAIT_L(8); BAR; WAIT_L(0); MMA(0, 0, At, B0); BAR; SCHED;
    LDB(B1, 1, 1); STAGE(SB(1, 0), b3);
    BAR; WAIT_L(0); MMA(0, 1, At, B1); BAR;
    LDA(At, 1, 1); STAGE(SA(1, 0), a3);
    BAR; WAIT_L(0); MMA(1, 0, At, B0); BAR; SCHED;
    STAGE(SB(1, 1), b3 + hstep);
    WAIT_V(6); BAR; MMA(1, 1, At, B1); BAR;
  }
  stamp(a, 4);
  {
    const bool bad = __hip_atomic_load(tmo, __ATOMIC_RELAXED, __HIP_MEMORY_SCOPE_AGENT) == call;
    const float nanv = __builtin_nanf("");
    const int row0 = pm * BM + wr * 64 + fr, col0 = pn * BM + wc * 32 + 4 * fq;
#pragma unroll
    for (int ai = 0; ai < 2; ++ai)
#pragma unroll
      for (int m = 0; m < 4; ++m) {
        float* rowp = a.out + (size_t)(row0 + ai * HALF + m * 16) * NL + col0;
#pragma unroll
        for (int bj = 0; bj < 2; ++bj)
#pragma unroll
          for (int n = 0; n < 2; ++n)
            *(f4*)(rowp + bj * HALF + n * 16) = bad ? (f4){nanv, nanv, nanv, nanv} : acc[ai][bj][m][n];
      }
  }
  WAIT_V(0);
  if (wr == 0) BAR;
  BAR;
  if (tid == 0) {
    const unsigned t = __hip_atomic_fetch_add((unsigned*)(myws + CTL_GDONE), 1u, __ATOMIC_RELAXED, __HIP_MEMORY_SCOPE_AGENT);
    if (t + 1u == call * (unsigned)NWG) {
      for (int p = 0; p < R; ++p)
        __hip_atomic_store((unsigned*)((char*)a.ws_all[p] + CTL_PDONE) + a.rank * 32, call, __ATOMIC_RELAXED, __HIP_MEMORY_SCOPE_SYSTEM);
    }
  }
  stamp(a, 5);
}
}

extern "C" void kernel_launch(int rank, int nranks, ncclComm_t comm, hipStream_t stream,
                              void* const* d_in, const int* in_sizes, int n_in, void* d_out, int out_size,
                              const int* in_dtypes, int out_dtype, void* const* d_out_all,
                              void* const* d_ws_all, size_t ws_bytes) {
  static bool init[R] = {};
  static hipStream_t side[R][R];
  static hipEvent_t ev_fork[R], ev_join[R][R];
  static void* wsp[R][R];
  if (!init[rank]) {
    (void)hipFuncSetAttribute((const void*)fused_kernel, hipFuncAttributeMaxDynamicSharedMemorySize, STAGE_BYTES);
    (void)hipEventCreateWithFlags(&ev_fork[rank], hipEventDisableTiming);
    for (int p = 0; p < R; ++p) {
      (void)hipStreamCreateWithFlags(&side[rank][p], hipStreamNonBlocking);
      (void)hipEventCreateWithFlags(&ev_join[rank][p], hipEventDisableTiming);
    }
    for (int p = 0; p < R; ++p) wsp[rank][p] = d_ws_all[p];
    init[rank] = true;
  }
  Args a;
  a.x = (const float*)d_in[0];
  a.w = (const float*)d_in[1];
  a.out = (float*)d_out;
  a.ws_all = d_ws_all;
  a.rank = rank;
  a.phase = 0;
  hipLaunchKernelGGL(fused_kernel, dim3(NWG), dim3(NTHR), STAGE_BYTES, stream, a);
  (void)hipEventRecord(ev_fork[rank], stream);
  char* me = (char*)wsp[rank][rank];
  for (int i = 1; i < R; ++i) {
    const int p = (rank + i) & (R - 1);
    hipStream_t s = side[rank][p];
    (void)hipStreamWaitEvent(s, ev_fork[rank], 0);
    char* peer = (char*)wsp[rank][p];
    for (int c = 0; c < NCH; ++c) {
      const size_t off = OFF_X + (size_t)rank * (NCH * CH_BYTES) + (size_t)c * CH_BYTES;
      (void)hipMemcpyAsync(peer + off, me + off, CH_BYTES, hipMemcpyDeviceToDevice, s);
      (void)hipMemcpyAsync(peer + CTL_FLAGS + (rank * 32 + c) * 4, me + CTL_SEQ, 4, hipMemcpyDeviceToDevice, s);
    }
    (void)hipEventRecord(ev_join[rank][p], s);
  }
  a.phase = 1;
  hipLaunchKernelGGL(fused_kernel, dim3(NWG), dim3(NTHR), STAGE_BYTES, stream, a);
  for (int i = 1; i < R; ++i) (void)hipStreamWaitEvent(stream, ev_join[rank][(rank + i) & (R - 1)], 0);
}
```

```cpp
#include <hip/hip_runtime.h>
#include <rccl/rccl.h>
#include <cstdio>

namespace {
#define LAS __attribute__((address_space(3)))
typedef _Float16 h8 __attribute__((ext_vector_type(8)));
typedef _Float16 h2 __attribute__((ext_vector_type(2)));
typedef float f4 __attribute__((ext_vector_type(4)));

constexpr int R = 8, M = 4096, K = 4096, NL = 4096, NG = 32768, MS = 512;
constexpr int BM = 256, BK = 64, HALF = 128, HTB = HALF * BK * 2, STAGE_BYTES = 8 * HTB;
constexpr int NWG = 256, NTHR = 512, NT = K / BK;
constexpr int LEAD = 16, EARLY = 16;
constexpr size_t TILE_BYTES = (size_t)BM * BK * 2;
constexpr int WUNITS = 512;
constexpr size_t OFF_X = 1ull << 20, XBYTES = (size_t)M * K * 2, OFF_WT = OFF_X + 2 * XBYTES;
constexpr int CTL_TICKET = 0, CTL_TMO = 256, CTL_WGRAB = 512, CTL_WDONE = 768, CTL_FLAGS = 4096;
constexpr unsigned long long DEADLINE_TICKS = 40000000ull;

__device__ __forceinline__ int lds_byte(int r, int c) {
  const int st = (r >> 4) * 2 + (c >> 5), rr = r & 15, cc = c & 31, ob = rr * 64 + cc * 2;
  return st * 1024 + (ob ^ (((ob >> 9) & 1) << 5));
}
__device__ __forceinline__ void stage_rc(int b, int& Rr, int& Cc) {
  const int st = b / 1024, sb = b % 1024, swz = sb ^ (((sb >> 9) & 1) << 5);
  Rr = (st >> 1) * 16 + swz / 64;
  Cc = (st & 1) * 32 + (swz % 64) / 2;
}

struct Args {
  const float* x;
  const float* w;
  float* out;
  void* const* ws_all;
  int rank;
};

#define stamp(a, slot)

__device__ __forceinline__ void load_piece(const Args& a, int kc, int h, h8 (&o)[4]) {
  const int tid = threadIdx.x;
  const int r0 = h * 256 + (tid >> 3), c = kc * 64 + (tid & 7) * 8;
  f4 v[8];
#pragma unroll
  for (int i = 0; i < 4; ++i) {
    const f4* src = (const f4*)(a.x + (size_t)(r0 + i * 64) * K + c);
    v[2 * i] = src[0]; v[2 * i + 1] = src[1];
  }
#pragma unroll
  for (int i = 0; i < 4; ++i)
#pragma unroll
    for (int j = 0; j < 4; ++j) { o[i][j] = (_Float16)v[2 * i][j]; o[i][4 + j] = (_Float16)v[2 * i + 1][j]; }
}

__device__ __forceinline__ char* piece_dst(const Args& a, unsigned call, int kc, int h, int d) {
  const int tid = threadIdx.x;
  const unsigned p = (unsigned)(a.rank + d) & (R - 1);
  return (char*)a.ws_all[p] + OFF_X + (size_t)(call & 1) * XBYTES + (size_t)((a.rank * 2 + h) * NT + kc) * TILE_BYTES + tid * 16;
}
constexpr size_t PIECE_STEP = 8192;

__device__ __forceinline__ void signal_piece(const Args& a, unsigned call, int kc, int h, int d0, int nd) {
  if ((int)threadIdx.x < nd) {
    __builtin_amdgcn_fence(__ATOMIC_RELEASE, "");
    asm volatile("s_waitcnt vmcnt(0)" ::: "memory");
    const unsigned p = (unsigned)(a.rank + d0 + (int)threadIdx.x) & (R - 1);
    unsigned* f = (unsigned*)((char*)a.ws_all[p] + CTL_FLAGS) + (a.rank * 2 + h) * 64 + kc;
    __hip_atomic_store(f, call, __ATOMIC_RELAXED, __HIP_MEMORY_SCOPE_SYSTEM);
  }
}

__device__ __forceinline__ void load_wtile(const Args& a, int tile, f4 (&v)[16]) {
  const int lane = threadIdx.x & 63;
  const int k0 = (tile >> 6) * 64, n0 = (tile & 63) * 64;
  const int c4 = lane & 15, q = lane >> 4;
  const float* src = a.w + (size_t)k0 * NG + (size_t)a.rank * NL + n0 + c4 * 4;
#pragma unroll
  for (int i = 0; i < 8; ++i) {
    v[2 * i] = *(const f4*)(src + (size_t)(i * 8 + q * 2) * NG);
    v[2 * i + 1] = *(const f4*)(src + (size_t)(i * 8 + q * 2 + 1) * NG);
  }
}

__device__ __forceinline__ void store_wtile(_Float16* wt, unsigned char* tl, int tile, const f4 (&v)[16]) {
  const int lane = threadIdx.x & 63;
  const int k0 = (tile >> 6) * 64, n0 = (tile & 63) * 64;
  const int c4 = lane & 15, q = lane >> 4;
#pragma unroll
  for (int i = 0; i < 8; ++i)
#pragma unroll
    for (int j = 0; j < 4; ++j) {
      h2 p; p[0] = (_Float16)v[2 * i][j]; p[1] = (_Float16)v[2 * i + 1][j];
      *(h2*)(tl + (c4 * 4 + j) * 136 + (i * 8 + q * 2) * 2) = p;
    }
  __builtin_amdgcn_s_waitcnt(0xc07f);
  __builtin_amdgcn_wave_barrier();
#pragma unroll
  for (int i = 0; i < 8; ++i) {
    const int n = i * 8 + (lane >> 3), k8 = lane & 7;
    typedef unsigned long long u64;
    typedef u64 u64x2 __attribute__((ext_vector_type(2)));
    const u64* s = (const u64*)(tl + n * 136 + k8 * 16);
    u64x2 v2; v2[0] = s[0]; v2[1] = s[1];
    *(u64x2*)((char*)wt + (size_t)((n0 >> 8) * NT + (k0 >> 6)) * TILE_BYTES + ((n0 & 255) + n) * 128 + k8 * 16) = v2;
  }
  __builtin_amdgcn_s_waitcnt(0xc07f);
  __builtin_amdgcn_wave_barrier();
}

__device__ __forceinline__ bool timed_out(unsigned* tmo, unsigned call, unsigned long long t0) {
  if (__hip_atomic_load(tmo, __ATOMIC_RELAXED, __HIP_MEMORY_SCOPE_AGENT) == call ||
      __builtin_amdgcn_s_memrealtime() - t0 > DEADLINE_TICKS) {
    __hip_atomic_store(tmo, call, __ATOMIC_RELAXED, __HIP_MEMORY_SCOPE_AGENT);
    return true;
  }
  return false;
}

__device__ __forceinline__ int wait_pieces(const unsigned* flags, unsigned* tmo, unsigned call, int need) {
  const int lane = threadIdx.x & 63;
  const unsigned long long t0 = __builtin_amdgcn_s_memrealtime();
  int ready;
  for (;;) {
    const unsigned f = __hip_atomic_load(flags + lane, __ATOMIC_RELAXED, __HIP_MEMORY_SCOPE_SYSTEM);
    const unsigned long long m = __ballot(f >= call);
    ready = (~m == 0ull) ? 64 : __builtin_ctzll(~m);
    if (ready > need) break;
    if (timed_out(tmo, call, t0)) { ready = 64; break; }
    __builtin_amdgcn_s_sleep(2);
  }
  asm volatile("" ::: "memory");
  return ready;
}

__global__ void __launch_bounds__(NTHR, 2) fused_kernel(Args a) {
  extern __shared__ __attribute__((aligned(16))) unsigned char shm[];
  LAS unsigned char* lds = (LAS unsigned char*)shm;
  stamp(a, 0);
  char* myws = (char*)a.ws_all[a.rank];
  unsigned* tmo = (unsigned*)(myws + CTL_TMO);
  const int tid = threadIdx.x, wid = __builtin_amdgcn_readfirstlane(tid >> 6), lane = tid & 63;

  unsigned call;
  {
    unsigned* sh = (unsigned*)shm;
    if (tid == 0) {
      const unsigned t = __hip_atomic_fetch_add((unsigned*)(myws + CTL_TICKET), 1u, __ATOMIC_RELAXED, __HIP_MEMORY_SCOPE_AGENT);
      sh[0] = t / gridDim.x + 1u;
      __builtin_amdgcn_fence(__ATOMIC_ACQUIRE, "");
      asm volatile("s_waitcnt vmcnt(0)" ::: "memory");
    }
    __syncthreads();
    call = sh[0];
    __syncthreads();
  }

  const int ekc = (int)blockIdx.x >> 4, eh = ((int)blockIdx.x >> 3) & 1, ed = (int)blockIdx.x & 7;
  const int job = (int)blockIdx.x;
  const int jkc = EARLY + (job >> 2), jh = (job >> 1) & 1, jd = (job & 1) * 4;
  const int jt = job < 4 * (NT - EARLY) ? ((jkc - LEAD) & ~1) : (1 << 20);

  {
    _Float16* wt = (_Float16*)(myws + OFF_WT);
    unsigned char* tl = shm + 64 + wid * (64 * 136);
    const int tile0 = (int)blockIdx.x * 16 + wid * 2;
    f4 va[16], vb[16];
    load_wtile(a, tile0, va);
    load_wtile(a, tile0 + 1, vb);
    {
      h8 eo[4];
      load_piece(a, ekc, eh, eo);
      char* dst = piece_dst(a, call, ekc, eh, ed);
#pragma unroll
      for (int j = 0; j < 4; ++j) *(h8*)(dst + j * PIECE_STEP) = eo[j];
    }
    store_wtile(wt, tl, tile0, va);
    store_wtile(wt, tl, tile0 + 1, vb);
    stamp(a, 6);
    asm volatile("s_waitcnt vmcnt(0)" ::: "memory");
    __syncthreads();
    stamp(a, 7);
    signal_piece(a, call, ekc, eh, ed, 1);
    if (tid == 0) {
      __builtin_amdgcn_fence(__ATOMIC_RELEASE, "agent");
      asm volatile("s_waitcnt vmcnt(0)" ::: "memory");
      unsigned* wd = (unsigned*)(myws + CTL_WDONE);
      __hip_atomic_fetch_add(wd, 1u, __ATOMIC_RELAXED, __HIP_MEMORY_SCOPE_AGENT);
      const unsigned long long t0 = __builtin_amdgcn_s_memrealtime();
      while (__hip_atomic_load(wd, __ATOMIC_RELAXED, __HIP_MEMORY_SCOPE_AGENT) < call * (unsigned)NWG) {
        if (timed_out(tmo, call, t0)) break;
        __builtin_amdgcn_s_sleep(1);
      }
      __builtin_amdgcn_fence(__ATOMIC_ACQUIRE, "agent");
      asm volatile("s_waitcnt vmcnt(0)" ::: "memory");
    }
    __syncthreads();
  }
  stamp(a, 2);

  const int wr = wid >> 2, wc = wid & 3, fr = lane & 15, fq = lane >> 4;
  int wg = blockIdx.x;
  wg = (wg & 7) * 32 + (wg >> 3);
  const int pm = (wg >> 5) * 2 + (wg & 1);
  const int pn = (wg & 31) >> 1;
  const char* Ab = myws + OFF_X + (size_t)(call & 1) * XBYTES;
  const char* Bb = myws + OFF_WT;
  const unsigned* flags = (const unsigned*)(myws + CTL_FLAGS) + pm * 64;

  unsigned voff[2];
#pragma unroll
  for (int i = 0; i < 2; ++i) { int Rr, Cc; stage_rc(tid * 16 + i * 8192, Rr, Cc); voff[i] = (unsigned)(Rr * BK + Cc) * 2u; }
  const size_t kstep = TILE_BYTES, hstep = TILE_BYTES / 2, tstep = TILE_BYTES * NT;
  const unsigned ldsw = (unsigned)wid * 1024u;
  const int aoff = lds_byte(wr * 64 + fr, fq * 8), boff = lds_byte(wc * 32 + fr, fq * 8);
#define SA(b, h) (((b) * 2 + (h)) * HTB)
#define SB(b, h) ((4 + (b) * 2 + (h)) * HTB)
#define STAGE(bufoff, gbase) do { _Pragma("unroll") for (int _i = 0; _i < 2; ++_i) \
    __builtin_amdgcn_global_load_lds((const unsigned*)((const char*)(gbase) + voff[_i]), (LAS unsigned*)(lds + (bufoff) + ldsw + _i * 8192), 16, 0, 0); } while (0)
#define LDA(dst, b, h) do { _Pragma("unroll") for (int m = 0; m < 4; ++m) _Pragma("unroll") for (int k = 0; k < 2; ++k) dst[m][k] = *(const LAS h8*)(lds + SA(b, h) + aoff + m * 2048 + k * 1024); } while (0)
#define LDB(dst, b, h) do { _Pragma("unroll") for (int n = 0; n < 2; ++n) _Pragma("unroll") for (int k = 0; k < 2; ++k) dst[n][k] = *(const LAS h8*)(lds + SB(b, h) + boff + n * 2048 + k * 1024); } while (0)
#define MMA(ai, bj, At, Bt) do { __builtin_amdgcn_s_setprio(1); _Pragma("unroll") for (int m = 0; m < 4; ++m) _Pragma("unroll") for (int n = 0; n < 2; ++n) _Pragma("unroll") for (int k = 0; k < 2; ++k) \
    acc[ai][bj][m][n] = __builtin_amdgcn_mfma_f32_16x16x32_f16(Bt[n][k], At[m][k], acc[ai][bj][m][n], 0, 0, 0); __builtin_amdgcn_s_setprio(0); } while (0)
#define WAIT_V(n) asm volatile("s_waitcnt vmcnt(" #n ")" ::: "memory")
#define WAIT_L(n) asm volatile("s_waitcnt lgkmcnt(" #n ")" ::: "memory")
#define BAR __builtin_amdgcn_s_barrier()
#define SCHED __builtin_amdgcn_sched_barrier(0)
#define PUSH(j) do { if (pushing) *(h8*)(pdst + (j) * PIECE_STEP) = po[j]; } while (0)
  f4 acc[2][2][4][2];
#pragma unroll
  for (int i = 0; i < 2; ++i)
#pragma unroll
    for (int j = 0; j < 2; ++j)
#pragma unroll
      for (int m = 0; m < 4; ++m)
#pragma unroll
        for (int n = 0; n < 2; ++n) acc[i][j][m][n] = (f4){0.f, 0.f, 0.f, 0.f};
  h8 At[4][2], B0[2][2], B1[2][2], po[4];
  const char* cA = Ab + (size_t)pm * tstep;
  const char* cB = Bb + (size_t)pn * tstep;
  int ready = wait_pieces(flags, tmo, call, 1);
  stamp(a, 3);
  STAGE(SB(0, 0), cB); STAGE(SA(0, 0), cA); STAGE(SB(0, 1), cB + hstep); STAGE(SA(0, 1), cA + hstep);
  if (wr == 1) BAR;
  WAIT_V(4); BAR;
  STAGE(SB(1, 0), cB + kstep); STAGE(SA(1, 0), cA + kstep); STAGE(SB(1, 1), cB + hstep + kstep);
  WAIT_V(6); BAR;
  for (int t = 0; t < NT; t += 2) {
    const bool last = (t == NT - 2);
    const char* a1 = cA + (size_t)(t + 1) * kstep;
    const char* a2 = last ? cA : cA + (size_t)(t + 2) * kstep;
    const char* b2 = last ? cB : cB + (size_t)(t + 2) * kstep;
    const char* a3 = a2 + kstep;
    const char* b3 = b2 + kstep;
    const int ph = (t - jt) >> 1;
    const bool pushing = ph >= 0 && ph < 4;
    char* pdst = nullptr;
    if (ph == 5) signal_piece(a, call, jkc, jh, jd, 4);
    if (ph == 0) load_piece(a, jkc, jh, po);
    if (pushing) pdst = piece_dst(a, call, jkc, jh, jd + ph);
    if (!last && ready <= t + 3) ready = wait_pieces(flags, tmo, call, t + 3);
    PUSH(0);
    LDB(B0, 0, 0); SCHED; LDA(At, 0, 0); STAGE(SA(1, 1), a1 + hstep);
    WAIT_L(8); BAR; WAIT_L(0); MMA(0, 0, At, B0); BAR; SCHED;
    LDB(B1, 0, 1); STAGE(SB(0, 0), b2);
    BAR; WAIT_L(0); MMA(0, 1, At, B1); BAR;
    PUSH(1);
    LDA(At, 0, 1); STAGE(SA(0, 0), a2);
    BAR; WAIT_L(0); MMA(1, 0, At, B0); BAR; SCHED;
    STAGE(SB(0, 1), b2 + hstep);
    WAIT_V(6); BAR; MMA(1, 1, At, B1); BAR;
    PUSH(2);
    LDB(B0, 1, 0); SCHED; LDA(At, 1, 0); STAGE(SA(0, 1), a2 + hstep);
    WAIT_L(8); BAR; WAIT_L(0); MMA(0, 0, At, B0); BAR; SCHED;
    LDB(B1, 1, 1); STAGE(SB(1, 0), b3);
    BAR; WAIT_L(0); MMA(0, 1, At, B1); BAR;
    PUSH(3);
    LDA(At, 1, 1); STAGE(SA(1, 0), a3);
    BAR; WAIT_L(0); MMA(1, 0, At, B0); BAR; SCHED;
    STAGE(SB(1, 1), b3 + hstep);
    WAIT_V(6); BAR; MMA(1, 1, At, B1); BAR;
  }
  stamp(a, 4);
  {
    const bool bad = __hip_atomic_load(tmo, __ATOMIC_RELAXED, __HIP_MEMORY_SCOPE_AGENT) == call;
    const float nanv = __builtin_nanf("");
    const int row0 = pm * BM + wr * 64 + fr, col0 = pn * BM + wc * 32 + 4 * fq;
#pragma unroll
    for (int ai = 0; ai < 2; ++ai)
#pragma unroll
      for (int m = 0; m < 4; ++m) {
        float* rowp = a.out + (size_t)(row0 + ai * HALF + m * 16) * NL + col0;
#pragma unroll
        for (int bj = 0; bj < 2; ++bj)
#pragma unroll
          for (int n = 0; n < 2; ++n)
            *(f4*)(rowp + bj * HALF + n * 16) = bad ? (f4){nanv, nanv, nanv, nanv} : acc[ai][bj][m][n];
      }
  }
  WAIT_V(0);
  if (wr == 0) BAR;
  BAR;
  stamp(a, 5);
}
}

extern "C" void kernel_launch(int rank, int nranks, ncclComm_t comm, hipStream_t stream,
                              void* const* d_in, const int* in_sizes, int n_in, void* d_out, int out_size,
                              const int* in_dtypes, int out_dtype, void* const* d_out_all,
                              void* const* d_ws_all, size_t ws_bytes) {
  static bool init = false;
  if (!init) {
    (void)hipFuncSetAttribute((const void*)fused_kernel, hipFuncAttributeMaxDynamicSharedMemorySize, STAGE_BYTES);
    init = true;
  }
  Args a;
  a.x = (const float*)d_in[0];
  a.w = (const float*)d_in[1];
  a.out = (float*)d_out;
  a.ws_all = d_ws_all;
  a.rank = rank;
  hipLaunchKernelGGL(fused_kernel, dim3(NWG), dim3(NTHR), STAGE_BYTES, stream, a);
}
```

```cpp
#include <hip/hip_runtime.h>
#include <rccl/rccl.h>
#include <cstdio>

namespace {
#define LAS __attribute__((address_space(3)))
typedef _Float16 h8 __attribute__((ext_vector_type(8)));
typedef _Float16 h2 __attribute__((ext_vector_type(2)));
typedef float f4 __attribute__((ext_vector_type(4)));

constexpr int R = 8, M = 4096, K = 4096, NL = 4096, NG = 32768, MS = 512;
constexpr int BM = 256, BK = 64, HALF = 128, HTB = HALF * BK * 2, STAGE_BYTES = 8 * HTB;
constexpr int NWG = 256, NTHR = 512, NT = K / BK;
constexpr int LEAD = 16, EARLY = 16, FIRST = 16;
constexpr size_t TILE_BYTES = (size_t)BM * BK * 2;
constexpr int WUNITS = 512;
constexpr size_t OFF_X = 1ull << 20, XBYTES = (size_t)M * K * 2, OFF_WT = OFF_X + 2 * XBYTES;
constexpr int CTL_TICKET = 0, CTL_TMO = 256, CTL_WGRAB = 512, CTL_WDONE = 768, CTL_FLAGS = 4096;
constexpr unsigned long long DEADLINE_TICKS = 40000000ull;

__device__ __forceinline__ int lds_byte(int r, int c) {
  const int st = (r >> 4) * 2 + (c >> 5), rr = r & 15, cc = c & 31, ob = rr * 64 + cc * 2;
  return st * 1024 + (ob ^ (((ob >> 9) & 1) << 5));
}
__device__ __forceinline__ void stage_rc(int b, int& Rr, int& Cc) {
  const int st = b / 1024, sb = b % 1024, swz = sb ^ (((sb >> 9) & 1) << 5);
  Rr = (st >> 1) * 16 + swz / 64;
  Cc = (st & 1) * 32 + (swz % 64) / 2;
}

struct Args {
  const float* x;
  const float* w;
  float* out;
  void* const* ws_all;
  int rank;
};

#define stamp(a, slot)

__device__ __forceinline__ void load_piece(const Args& a, int kc, int h, h8 (&o)[4]) {
  const int tid = threadIdx.x;
  const int r0 = h * 256 + (tid >> 3), c = kc * 64 + (tid & 7) * 8;
  f4 v[8];
#pragma unroll
  for (int i = 0; i < 4; ++i) {
    const f4* src = (const f4*)(a.x + (size_t)(r0 + i * 64) * K + c);
    v[2 * i] = src[0]; v[2 * i + 1] = src[1];
  }
#pragma unroll
  for (int i = 0; i < 4; ++i)
#pragma unroll
    for (int j = 0; j < 4; ++j) { o[i][j] = (_Float16)v[2 * i][j]; o[i][4 + j] = (_Float16)v[2 * i + 1][j]; }
}

__device__ __forceinline__ char* piece_dst(const Args& a, unsigned call, int kc, int h, int d) {
  const int tid = threadIdx.x;
  const unsigned p = (unsigned)(a.rank + d) & (R - 1);
  return (char*)a.ws_all[p] + OFF_X + (size_t)(call & 1) * XBYTES + (size_t)((a.rank * 2 + h) * NT + kc) * TILE_BYTES + tid * 16;
}
constexpr size_t PIECE_STEP = 8192;

__device__ __forceinline__ void st16_sys(char* p, h8 v) {
  asm volatile("global_store_dwordx4 %0, %1, off sc0 sc1\n\ts_nop 1" ::"v"(p), "v"(v) : "memory");
}

__device__ __forceinline__ void signal_piece(const Args& a, unsigned call, int kc, int h, int d0, int nd) {
  if ((int)threadIdx.x < nd) {
    asm volatile("s_waitcnt vmcnt(0)" ::: "memory");
    const unsigned p = (unsigned)(a.rank + d0 + (int)threadIdx.x) & (R - 1);
    unsigned* f = (unsigned*)((char*)a.ws_all[p] + CTL_FLAGS) + (a.rank * 2 + h) * 64 + kc;
    __hip_atomic_store(f, call, __ATOMIC_RELAXED, __HIP_MEMORY_SCOPE_SYSTEM);
  }
}

__device__ __forceinline__ void load_wtile(const Args& a, int tile, f4 (&v)[16]) {
  const int lane = threadIdx.x & 63;
  const int k0 = (tile >> 6) * 64, n0 = (tile & 63) * 64;
  const int c4 = lane & 15, q = lane >> 4;
  const float* src = a.w + (size_t)k0 * NG + (size_t)a.rank * NL + n0 + c4 * 4;
#pragma unroll
  for (int i = 0; i < 8; ++i) {
    v[2 * i] = *(const f4*)(src + (size_t)(i * 8 + q * 2) * NG);
    v[2 * i + 1] = *(const f4*)(src + (size_t)(i * 8 + q * 2 + 1) * NG);
  }
}

__device__ __forceinline__ void store_wtile(_Float16* wt, unsigned char* tl, int tile, const f4 (&v)[16]) {
  const int lane = threadIdx.x & 63;
  const int k0 = (tile >> 6) * 64, n0 = (tile & 63) * 64;
  const int c4 = lane & 15, q = lane >> 4;
#pragma unroll
  for (int i = 0; i < 8; ++i)
#pragma unroll
    for (int j = 0; j < 4; ++j) {
      h2 p; p[0] = (_Float16)v[2 * i][j]; p[1] = (_Float16)v[2 * i + 1][j];
      *(h2*)(tl + (c4 * 4 + j) * 136 + (i * 8 + q * 2) * 2) = p;
    }
  __builtin_amdgcn_s_waitcnt(0xc07f);
  __builtin_amdgcn_wave_barrier();
#pragma unroll
  for (int i = 0; i < 8; ++i) {
    const int n = i * 8 + (lane >> 3), k8 = lane & 7;
    typedef unsigned long long u64;
    typedef u64 u64x2 __attribute__((ext_vector_type(2)));
    const u64* s = (const u64*)(tl + n * 136 + k8 * 16);
    u64x2 v2; v2[0] = s[0]; v2[1] = s[1];
    *(u64x2*)((char*)wt + (size_t)((n0 >> 8) * NT + (k0 >> 6)) * TILE_BYTES + ((n0 & 255) + n) * 128 + k8 * 16) = v2;
  }
  __builtin_amdgcn_s_waitcnt(0xc07f);
  __builtin_amdgcn_wave_barrier();
}

__device__ __forceinline__ bool timed_out(unsigned* tmo, unsigned call, unsigned long long t0) {
  if (__hip_atomic_load(tmo, __ATOMIC_RELAXED, __HIP_MEMORY_SCOPE_AGENT) == call ||
      __builtin_amdgcn_s_memrealtime() - t0 > DEADLINE_TICKS) {
    __hip_atomic_store(tmo, call, __ATOMIC_RELAXED, __HIP_MEMORY_SCOPE_AGENT);
    return true;
  }
  return false;
}

__device__ __forceinline__ int wait_pieces(const unsigned* flags, unsigned* tmo, unsigned call, int need) {
  const int lane = threadIdx.x & 63;
  const unsigned long long t0 = __builtin_amdgcn_s_memrealtime();
  int ready;
  for (;;) {
    const unsigned f = __hip_atomic_load(flags + lane, __ATOMIC_RELAXED, __HIP_MEMORY_SCOPE_SYSTEM);
    const unsigned long long m = __ballot(f >= call);
    ready = (~m == 0ull) ? 64 : __builtin_ctzll(~m);
    if (ready > need) break;
    if (timed_out(tmo, call, t0)) { ready = 64; break; }
    __builtin_amdgcn_s_sleep(2);
  }
  asm volatile("" ::: "memory");
  return ready;
}

__global__ void __launch_bounds__(NTHR, 2) fused_kernel(Args a) {
  extern __shared__ __attribute__((aligned(16))) unsigned char shm[];
  LAS unsigned char* lds = (LAS unsigned char*)shm;
  stamp(a, 0);
  char* myws = (char*)a.ws_all[a.rank];
  unsigned* tmo = (unsigned*)(myws + CTL_TMO);
  const int tid = threadIdx.x, wid = __builtin_amdgcn_readfirstlane(tid >> 6), lane = tid & 63;

  unsigned call;
  {
    unsigned* sh = (unsigned*)shm;
    if (tid == 0) {
      const unsigned t = __hip_atomic_fetch_add((unsigned*)(myws + CTL_TICKET), 1u, __ATOMIC_RELAXED, __HIP_MEMORY_SCOPE_AGENT);
      sh[0] = t / gridDim.x + 1u;
      __builtin_amdgcn_fence(__ATOMIC_ACQUIRE, "");
      asm volatile("s_waitcnt vmcnt(0)" ::: "memory");
    }
    __syncthreads();
    call = sh[0];
    __syncthreads();
  }

  const int ekc = (int)blockIdx.x >> 4, eh = ((int)blockIdx.x >> 3) & 1, ed = (int)blockIdx.x & 7;
  const int job = (int)blockIdx.x;
  const int jkc = EARLY + (job >> 2), jh = (job >> 1) & 1, jd = (job & 1) * 4;
  const int jt = job < 4 * (NT - EARLY) ? ((jkc - LEAD) & ~1) : (1 << 20);

  {
    _Float16* wt = (_Float16*)(myws + OFF_WT);
    unsigned char* tl = shm + 64 + wid * (64 * 136);
    const int tile0 = (int)blockIdx.x * 16 + wid * 2;
    f4 va[16], vb[16];
    load_wtile(a, tile0, va);
    load_wtile(a, tile0 + 1, vb);
    if (ekc < FIRST) {
      h8 eo[4];
      load_piece(a, ekc, eh, eo);
      char* dst = piece_dst(a, call, ekc, eh, ed);
#pragma unroll
      for (int j = 0; j < 4; ++j) st16_sys(dst + j * PIECE_STEP, eo[j]);
    }
    store_wtile(wt, tl, tile0, va);
    store_wtile(wt, tl, tile0 + 1, vb);
    stamp(a, 6);
    asm volatile("s_waitcnt vmcnt(0)" ::: "memory");
    __syncthreads();
    stamp(a, 7);
    if (ekc < FIRST) signal_piece(a, call, ekc, eh, ed, 1);
    if (tid == 0) {
      __builtin_amdgcn_fence(__ATOMIC_RELEASE, "agent");
      asm volatile("s_waitcnt vmcnt(0)" ::: "memory");
      unsigned* wd = (unsigned*)(myws + CTL_WDONE);
      __hip_atomic_fetch_add(wd, 1u, __ATOMIC_RELAXED, __HIP_MEMORY_SCOPE_AGENT);
      const unsigned long long t0 = __builtin_amdgcn_s_memrealtime();
      while (__hip_atomic_load(wd, __ATOMIC_RELAXED, __HIP_MEMORY_SCOPE_AGENT) < call * (unsigned)NWG) {
        if (timed_out(tmo, call, t0)) break;
        __builtin_amdgcn_s_sleep(1);
      }
      __builtin_amdgcn_fence(__ATOMIC_ACQUIRE, "agent");
      asm volatile("s_waitcnt vmcnt(0)" ::: "memory");
    }
    __syncthreads();
    if (ekc >= FIRST) {
      h8 eo[4];
      load_piece(a, ekc, eh, eo);
      char* dst = piece_dst(a, call, ekc, eh, ed);
#pragma unroll
      for (int j = 0; j < 4; ++j) st16_sys(dst + j * PIECE_STEP, eo[j]);
    }
  }
  stamp(a, 2);

  const int wr = wid >> 2, wc = wid & 3, fr = lane & 15, fq = lane >> 4;
  int wg = blockIdx.x;
  wg = (wg & 7) * 32 + (wg >> 3);
  const int pm = (wg >> 5) * 2 + (wg & 1);
  const int pn = (wg & 31) >> 1;
  const char* Ab = myws + OFF_X + (size_t)(call & 1) * XBYTES;
  const char* Bb = myws + OFF_WT;
  const unsigned* flags = (const unsigned*)(myws + CTL_FLAGS) + pm * 64;

  unsigned voff[2];
#pragma unroll
  for (int i = 0; i < 2; ++i) { int Rr, Cc; stage_rc(tid * 16 + i * 8192, Rr, Cc); voff[i] = (unsigned)(Rr * BK + Cc) * 2u; }
  const size_t kstep = TILE_BYTES, hstep = TILE_BYTES / 2, tstep = TILE_BYTES * NT;
  const unsigned ldsw = (unsigned)wid * 1024u;
  const int aoff = lds_byte(wr * 64 + fr, fq * 8), boff = lds_byte(wc * 32 + fr, fq * 8);
#define SA(b, h) (((b) * 2 + (h)) * HTB)
#define SB(b, h) ((4 + (b) * 2 + (h)) * HTB)
#define STAGE(bufoff, gbase) do { _Pragma("unroll") for (int _i = 0; _i < 2; ++_i) \
    __builtin_amdgcn_global_load_lds((const unsigned*)((const char*)(gbase) + voff[_i]), (LAS unsigned*)(lds + (bufoff) + ldsw + _i * 8192), 16, 0, 0); } while (0)
#define LDA(dst, b, h) do { _Pragma("unroll") for (int m = 0; m < 4; ++m) _Pragma("unroll") for (int k = 0; k < 2; ++k) dst[m][k] = *(const LAS h8*)(lds + SA(b, h) + aoff + m * 2048 + k * 1024); } while (0)
#define LDB(dst, b, h) do { _Pragma("unroll") for (int n = 0; n < 2; ++n) _Pragma("unroll") for (int k = 0; k < 2; ++k) dst[n][k] = *(const LAS h8*)(lds + SB(b, h) + boff + n * 2048 + k * 1024); } while (0)
#define MMA(ai, bj, At, Bt) do { __builtin_amdgcn_s_setprio(1); _Pragma("unroll") for (int m = 0; m < 4; ++m) _Pragma("unroll") for (int n = 0; n < 2; ++n) _Pragma("unroll") for (int k = 0; k < 2; ++k) \
    acc[ai][bj][m][n] = __builtin_amdgcn_mfma_f32_16x16x32_f16(Bt[n][k], At[m][k], acc[ai][bj][m][n], 0, 0, 0); __builtin_amdgcn_s_setprio(0); } while (0)
#define WAIT_V(n) asm volatile("s_waitcnt vmcnt(" #n ")" ::: "memory")
#define WAIT_L(n) asm volatile("s_waitcnt lgkmcnt(" #n ")" ::: "memory")
#define BAR __builtin_amdgcn_s_barrier()
#define SCHED __builtin_amdgcn_sched_barrier(0)
#define PUSH(j) do { if (pushing) st16_sys(pdst + (j) * PIECE_STEP, po[j]); } while (0)
  f4 acc[2][2][4][2];
#pragma unroll
  for (int i = 0; i < 2; ++i)
#pragma unroll
    for (int j = 0; j < 2; ++j)
#pragma unroll
      for (int m = 0; m < 4; ++m)
#pragma unroll
        for (int n = 0; n < 2; ++n) acc[i][j][m][n] = (f4){0.f, 0.f, 0.f, 0.f};
  h8 At[4][2], B0[2][2], B1[2][2], po[4];
  const char* cA = Ab + (size_t)pm * tstep;
  const char* cB = Bb + (size_t)pn * tstep;
  int ready = wait_pieces(flags, tmo, call, 1);
  stamp(a, 3);
  STAGE(SB(0, 0), cB); STAGE(SA(0, 0), cA); STAGE(SB(0, 1), cB + hstep); STAGE(SA(0, 1), cA + hstep);
  if (wr == 1) BAR;
  WAIT_V(4); BAR;
  STAGE(SB(1, 0), cB + kstep); STAGE(SA(1, 0), cA + kstep); STAGE(SB(1, 1), cB + hstep + kstep);
  WAIT_V(6); BAR;
  for (int t = 0; t < NT; t += 2) {
    const bool last = (t == NT - 2);
    const char* a1 = cA + (size_t)(t + 1) * kstep;
    const char* a2 = last ? cA : cA + (size_t)(t + 2) * kstep;
    const char* b2 = last ? cB : cB + (size_t)(t + 2) * kstep;
    const char* a3 = a2 + kstep;
    const char* b3 = b2 + kstep;
    const int ph = (t - jt) >> 1;
    const bool pushing = ph >= 0 && ph < 4;
    char* pdst = nullptr;
    if (t == 2 && ekc >= FIRST) signal_piece(a, call, ekc, eh, ed, 1);
    if (ph == 5) signal_piece(a, call, jkc, jh, jd, 4);
    if (ph == 0) load_piece(a, jkc, jh, po);
    if (pushing) pdst = piece_dst(a, call, jkc, jh, jd + ph);
    if (!last && ready <= t + 3) ready = wait_pieces(flags, tmo, call, t + 3);
    PUSH(0);
    LDB(B0, 0, 0); SCHED; LDA(At, 0, 0); STAGE(SA(1, 1), a1 + hstep);
    WAIT_L(8); BAR; WAIT_L(0); MMA(0, 0, At, B0); BAR; SCHED;
    LDB(B1, 0, 1); STAGE(SB(0, 0), b2);
    BAR; WAIT_L(0); MMA(0, 1, At, B1); BAR;
    PUSH(1);
    LDA(At, 0, 1); STAGE(SA(0, 0), a2);
    BAR; WAIT_L(0); MMA(1, 0, At, B0); BAR; SCHED;
    STAGE(SB(0, 1), b2 + hstep);
    WAIT_V(6); BAR; MMA(1, 1, At, B1); BAR;
    PUSH(2);
    LDB(B0, 1, 0); SCHED; LDA(At, 1, 0); STAGE(SA(0, 1), a2 + hstep);
    WAIT_L(8); BAR; WAIT_L(0); MMA(0, 0, At, B0); BAR; SCHED;
    LDB(B1, 1, 1); STAGE(SB(1, 0), b3);
    BAR; WAIT_L(0); MMA(0, 1, At, B1); BAR;
    PUSH(3);
    LDA(At, 1, 1); STAGE(SA(1, 0), a3);
    BAR; WAIT_L(0); MMA(1, 0, At, B0); BAR; SCHED;
    STAGE(SB(1, 1), b3 + hstep);
    WAIT_V(6); BAR; MMA(1, 1, At, B1); BAR;
  }
  stamp(a, 4);
  {
    const bool bad = __hip_atomic_load(tmo, __ATOMIC_RELAXED, __HIP_MEMORY_SCOPE_AGENT) == call;
    const float nanv = __builtin_nanf("");
    const int row0 = pm * BM + wr * 64 + fr, col0 = pn * BM + wc * 32 + 4 * fq;
#pragma unroll
    for (int ai = 0; ai < 2; ++ai)
#pragma unroll
      for (int m = 0; m < 4; ++m) {
        float* rowp = a.out + (size_t)(row0 + ai * HALF + m * 16) * NL + col0;
#pragma unroll
        for (int bj = 0; bj < 2; ++bj)
#pragma unroll
          for (int n = 0; n < 2; ++n)
            *(f4*)(rowp + bj * HALF + n * 16) = bad ? (f4){nanv, nanv, nanv, nanv} : acc[ai][bj][m][n];
      }
  }
  WAIT_V(0);
  if (wr == 0) BAR;
  BAR;
  stamp(a, 5);
}
}

extern "C" void kernel_launch(int rank, int nranks, ncclComm_t comm, hipStream_t stream,
                              void* const* d_in, const int* in_sizes, int n_in, void* d_out, int out_size,
                              const int* in_dtypes, int out_dtype, void* const* d_out_all,
                              void* const* d_ws_all, size_t ws_bytes) {
  static bool init = false;
  if (!init) {
    (void)hipFuncSetAttribute((const void*)fused_kernel, hipFuncAttributeMaxDynamicSharedMemorySize, STAGE_BYTES);
    init = true;
  }
  Args a;
  a.x = (const float*)d_in[0];
  a.w = (const float*)d_in[1];
  a.out = (float*)d_out;
  a.ws_all = d_ws_all;
  a.rank = rank;
  hipLaunchKernelGGL(fused_kernel, dim3(NWG), dim3(NTHR), STAGE_BYTES, stream, a);
}
```
